```python
import numpy as np

kernel_name = "gpu_gemm_ar_m4096_k4096_n4096_bf16_q8"


ABSMAX_RTOL = 0.02

COMPUTE_DTYPE = "bfloat16"
EPILOGUE = "quant_int8"
IN_QUANT = None
OUT_QUANT = "int8"
IN_DTYPES = {"x": "bfloat16", "w_mat": "bfloat16"}
WORLD_SIZE = 8

def _to_bf16(x):
    u = np.ascontiguousarray(x, dtype=np.float32).view(np.uint32)
    return ((u + 0x7FFF + ((u >> 16) & 1)) & 0xFFFF0000).view(np.float32)

def setup_inputs():

    x = _to_bf16(np.random.randn(4096, 4096).astype(np.float32))
    w_mat = _to_bf16((np.random.randn(4096, 4096) / np.sqrt(4096)).astype(np.float32))
    return {"x": x, "w_mat": w_mat}

def _quant_dequant(y, amax):
    scale = amax / 127.0
    q = np.clip(np.round(y / scale), -127.0, 127.0)
    return (q * scale).astype(np.float32)
def _epilogue(y):
    return _quant_dequant(y, np.abs(y).max())

def reference(x, w_mat):
    y = (x @ w_mat).astype(np.float32)
    return _epilogue(y)
```

```cpp
#include <hip/hip_runtime.h>
#include <rccl/rccl.h>
#include <cstdio>
#include <cstdint>

namespace {

typedef short bf16x8 __attribute__((ext_vector_type(8)));
typedef float f32x4 __attribute__((ext_vector_type(4)));
typedef unsigned v4u __attribute__((ext_vector_type(4)));
typedef unsigned long long u64;

constexpr int NR = 8;
constexpr int MD = 4096, ND = 4096;
constexpr int KSH = 512;
constexpr int TROWS = 1024, TCOLS = 2048;
constexpr int NC = 8;
constexpr int NCOMM = 128;
constexpr int NGEMM = 256;
constexpr int GRID = NCOMM + NGEMM;
constexpr int NTICK = 16;
constexpr int WG_PER_TICK = GRID / NTICK;

constexpr size_t OFF_TICK = 0;
constexpr size_t OFF_TMO = 4096;
constexpr size_t OFF_P1F = 8192;
constexpr size_t OFF_AMX = OFF_P1F + 32768;
constexpr size_t OFF_OUTF = OFF_AMX + 16384;
constexpr size_t OFF_A = 1u << 20;
constexpr size_t OFF_B = OFF_A + (8u << 20);
constexpr size_t WS_NEED = OFF_B + (16u << 20);

constexpr u64 DEADLINE_TICKS = 100000000ull;

struct Params {
  const unsigned short* x;
  const unsigned short* w;
  void* const* out_all;
  void* const* ws_all;
  int rank;
};

__device__ __forceinline__ void st16_sys(void* p, v4u v) {
  asm volatile("global_store_dwordx4 %0, %1, off sc0 sc1\n\ts_nop 2" ::"v"(p), "v"(v) : "memory");
}
__device__ __forceinline__ void st4_sys(void* p, unsigned v) {
  __hip_atomic_store((unsigned*)p, v, __ATOMIC_RELAXED, __HIP_MEMORY_SCOPE_SYSTEM);
}
#define TS(i) do {} while (0)
#define WAITV(n) asm volatile("s_waitcnt vmcnt(" #n ")" ::: "memory")

__device__ __forceinline__ bool timed_out(unsigned* tmo, unsigned call, u64 t0) {
  if (__hip_atomic_load(tmo, __ATOMIC_RELAXED, __HIP_MEMORY_SCOPE_AGENT) == call ||
      __builtin_amdgcn_s_memrealtime() - t0 > DEADLINE_TICKS) {
    __hip_atomic_store(tmo, call, __ATOMIC_RELAXED, __HIP_MEMORY_SCOPE_AGENT);
    return true;
  }
  return false;
}

__device__ __forceinline__ void wait_p1(char* ws, int p, int c, unsigned call, int lane) {
  const u64* fp = (const u64*)(ws + OFF_P1F) + ((p * NC + c) * 64 + lane);
  u64 v = __hip_atomic_load(fp, __ATOMIC_RELAXED, __HIP_MEMORY_SCOPE_SYSTEM);
  bool ok = ((unsigned)v >= call) && ((unsigned)(v >> 32) >= call);
  if (!__all(ok)) {
    unsigned* tmo = (unsigned*)(ws + OFF_TMO);
    u64 t0 = __builtin_amdgcn_s_memrealtime();
    for (;;) {
      __builtin_amdgcn_s_sleep(2);
      v = __hip_atomic_load(fp, __ATOMIC_RELAXED, __HIP_MEMORY_SCOPE_SYSTEM);
      ok = ((unsigned)v >= call) && ((unsigned)(v >> 32) >= call);
      if (__all(ok)) break;
      if (timed_out(tmo, call, t0)) break;
    }
  }
  asm volatile("" ::: "memory");
}

__global__ __launch_bounds__(128) void gemm_ar_q8_fused(Params prm) {
  __shared__ __attribute__((aligned(16))) unsigned char smem[64 * 144 + 64];
  const int tid = threadIdx.x, lane = tid & 63, wv = tid >> 6;
  const int self = prm.rank;
#define WSP(p) ((char*)prm.ws_all[(p)])
  char* ws = WSP(self);

  unsigned* sh_u = (unsigned*)(smem + 64 * 144);
  if (tid == 0) {
    unsigned t = __hip_atomic_fetch_add((unsigned*)(ws + OFF_TICK + (blockIdx.x % NTICK) * 128), 1u,
                                        __ATOMIC_RELAXED, __HIP_MEMORY_SCOPE_AGENT);
    sh_u[0] = t / WG_PER_TICK + 1;
  }
  if (tid >= 1 && tid <= 10) sh_u[tid] = 0;
  __syncthreads();
  const unsigned call = sh_u[0];
  TS(0);
  unsigned* tmo = (unsigned*)(ws + OFF_TMO);

  if (blockIdx.x < NCOMM) {
    const int v = blockIdx.x * 2 + wv;
    const int g = lane >> 4, r = lane & 15;
    v4u xr[NC][2];
    unsigned wr_[NC][2][4];
    const unsigned short* xs = prm.x + (size_t)(16 * v + r) * KSH + 8 * g;
    const unsigned short* wsrc = prm.w + (size_t)(8 * g) * ND + 16 * v + r;
#pragma unroll
    for (int c = 0; c < NC; ++c) {
#pragma unroll
      for (int ks = 0; ks < 2; ++ks) {
        xr[c][ks] = *(const v4u*)(xs + 64 * c + 32 * ks);
#pragma unroll
        for (int jj = 0; jj < 4; ++jj) {
          unsigned lo = wsrc[(size_t)(64 * c + 32 * ks + 2 * jj) * ND];
          unsigned hi = wsrc[(size_t)(64 * c + 32 * ks + 2 * jj + 1) * ND];
          wr_[c][ks][jj] = lo | (hi << 16);
        }
      }
    }
    const int di = v >> 6, dj = v >> 7;
    char* xd[2];
    char* wd[4];
#pragma unroll
    for (int q = 0; q < 2; ++q) xd[q] = WSP(2 * di + ((q + v) & 1)) + OFF_A + (size_t)(v & 63) * 1024 + lane * 16;
#pragma unroll
    for (int q = 0; q < 4; ++q) wd[q] = WSP(dj + 2 * ((q + v) & 3)) + OFF_B + (size_t)(v & 127) * 1024 + lane * 16;
    char* fl = WSP(lane & 7);
    unsigned* flbase = (unsigned*)(fl + OFF_P1F) + (size_t)self * NC * NCOMM + blockIdx.x;
#pragma unroll
    for (int c = 0; c <= NC; ++c) {
      if (c < NC) {
#pragma unroll
        for (int ks = 0; ks < 2; ++ks) {
          const size_t oa = (size_t)((self * NC + c) * 2 + ks) * 64 * 1024;
          const size_t ob = (size_t)((self * NC + c) * 2 + ks) * 128 * 1024;
          v4u wv4 = {wr_[c][ks][0], wr_[c][ks][1], wr_[c][ks][2], wr_[c][ks][3]};
#pragma unroll
          for (int q = 0; q < 4; ++q) st16_sys(wd[q] + ob, wv4);
#pragma unroll
          for (int q = 0; q < 2; ++q) st16_sys(xd[q] + oa, xr[c][ks]);
        }
      }
      if (c > 0) {
        if (c < NC) WAITV(12); else WAITV(0);
        unsigned old = 0;
        if (lane == 0) old = atomicAdd(&sh_u[c], 1u);
        old = __builtin_amdgcn_readfirstlane(old);
        if (old == 1 && lane < NR) st4_sys(flbase + (size_t)(c - 1) * NCOMM, call);
        if (c == 1) TS(1);
      }
    }
    TS(2);
    return;
  }

  const int gi = blockIdx.x - NCOMM;
  const int xcd = gi & 7, sub = gi >> 3;
  const int tm = (xcd >> 2) * 8 + (sub & 7);
  const int tn = (xcd & 3) * 4 + (sub >> 3);
  const int ri = self >> 1, cj = self & 1;

  __builtin_amdgcn_fence(__ATOMIC_ACQUIRE, "");

  f32x4 acc[4][4];
#pragma unroll
  for (int a = 0; a < 4; ++a)
#pragma unroll
    for (int b = 0; b < 4; ++b) acc[a][b] = (f32x4){0.f, 0.f, 0.f, 0.f};

  const char* abase = ws + OFF_A + (size_t)(tm * 4) * 1024 + lane * 16;
  const char* bbase = ws + OFF_B + (size_t)(tn * 8 + wv * 4) * 1024 + lane * 16;

  bf16x8 fa[4][4], fb[4][4];
  auto gaddr = [&](int q, size_t& oa, size_t& ob) {
    const int s = q >> 1, ks = q & 1, c = s >> 3, p = (self + (s & 7)) & 7;
    const size_t blk = (size_t)((p * NC + c) * 2 + ks);
    oa = blk * 64 * 1024;
    ob = blk * 128 * 1024;
  };
  auto issue = [&](int q, bf16x8 (&A)[4], bf16x8 (&B)[4]) {
    size_t oa, ob;
    gaddr(q, oa, ob);
#pragma unroll
    for (int i = 0; i < 4; ++i) A[i] = *(const bf16x8*)(abase + oa + i * 1024);
#pragma unroll
    for (int i = 0; i < 4; ++i) B[i] = *(const bf16x8*)(bbase + ob + i * 1024);
  };
  auto waitstep = [&](int q) {
    const int s = q >> 1, c = s >> 3, p = (self + (s & 7)) & 7;
    wait_p1(ws, p, c, call, lane);
  };
  auto mma = [&](bf16x8 (&A)[4], bf16x8 (&B)[4]) {
#pragma unroll
    for (int nt = 0; nt < 4; ++nt)
#pragma unroll
      for (int mt = 0; mt < 4; ++mt)
        acc[nt][mt] = __builtin_amdgcn_mfma_f32_16x16x32_bf16(B[nt], A[mt], acc[nt][mt], 0, 0, 0);
  };

  waitstep(0);
  TS(1);
  issue(0, fa[0], fb[0]);
  issue(1, fa[1], fb[1]);
  waitstep(2);
  issue(2, fa[2], fb[2]);
  for (int q = 0; q < 128; q += 4) {
    if (q + 3 < 128) issue(q + 3, fa[3], fb[3]);
    mma(fa[0], fb[0]);
    if (q + 4 < 128) { waitstep(q + 4); issue(q + 4, fa[0], fb[0]); }
    mma(fa[1], fb[1]);
    if (q + 5 < 128) issue(q + 5, fa[1], fb[1]);
    mma(fa[2], fb[2]);
    if (q + 6 < 128) { waitstep(q + 6); issue(q + 6, fa[2], fb[2]); }
    mma(fa[3], fb[3]);
  }

  TS(2);
  float am = 0.f;
#pragma unroll
  for (int a = 0; a < 4; ++a)
#pragma unroll
    for (int b = 0; b < 4; ++b)
#pragma unroll
      for (int e = 0; e < 4; ++e) am = fmaxf(am, fabsf(acc[a][b][e]));
#pragma unroll
  for (int o = 32; o > 0; o >>= 1) am = fmaxf(am, __shfl_xor(am, o));
  float* sh_f = (float*)(sh_u + 12);
  if (lane == 0) sh_f[wv] = am;
  __syncthreads();
  if (wv == 0) {
    const float wam = fmaxf(sh_f[0], sh_f[1]);
    if (lane < NR) {
      u64 gr = ((u64)call << 32) | (u64)__float_as_uint(wam);
      __hip_atomic_store((u64*)(WSP(lane) + OFF_AMX) + (self * NGEMM + gi), gr, __ATOMIC_RELAXED,
                         __HIP_MEMORY_SCOPE_SYSTEM);
    }
    const u64* gp = (const u64*)(ws + OFF_AMX) + lane;
    float gm = 0.f;
    u64 t0 = __builtin_amdgcn_s_memrealtime();
    for (int k = 0; k < NR * NGEMM / 64; ++k) {
      u64 v = __hip_atomic_load(gp + k * 64, __ATOMIC_RELAXED, __HIP_MEMORY_SCOPE_SYSTEM);
      while (!__all((unsigned)(v >> 32) == call)) {
        if (timed_out(tmo, call, t0)) break;
        __builtin_amdgcn_s_sleep(1);
        v = __hip_atomic_load(gp + k * 64, __ATOMIC_RELAXED, __HIP_MEMORY_SCOPE_SYSTEM);
      }
      gm = fmaxf(gm, __uint_as_float((unsigned)v));
    }
#pragma unroll
    for (int o = 32; o > 0; o >>= 1) gm = fmaxf(gm, __shfl_xor(gm, o));
    if (lane == 0) sh_f[2] = gm;
  }
  __syncthreads();
  TS(3);
  const float amax = sh_f[2];
  const bool bad = (__hip_atomic_load(tmo, __ATOMIC_RELAXED, __HIP_MEMORY_SCOPE_AGENT) == call);
  const float scale = amax / 127.0f;

  {
    const int g = lane >> 4, mm = lane & 15;
#pragma unroll
    for (int nt = 0; nt < 4; ++nt)
#pragma unroll
      for (int mt = 0; mt < 4; ++mt) {
        unsigned pk = 0;
#pragma unroll
        for (int e = 0; e < 4; ++e) {
          float qf = (scale > 0.f) ? rintf(acc[nt][mt][e] / scale) : 0.f;
          qf = fminf(fmaxf(qf, -127.f), 127.f);
          int qi = bad ? -128 : (int)qf;
          pk |= ((unsigned)qi & 0xffu) << (8 * e);
        }
        *(unsigned*)(smem + (mt * 16 + mm) * 144 + wv * 64 + nt * 16 + g * 4) = pk;
      }
  }
  __syncthreads();
  {
    const int row0 = ri * TROWS + tm * 64, col0 = cj * TCOLS + tn * 128;
    const int tr = tid >> 3, tc = tid & 7;
    v4u o4[4];
#pragma unroll
    for (int ps = 0; ps < 4; ++ps) o4[ps] = *(const v4u*)(smem + (ps * 16 + tr) * 144 + tc * 16);
    const size_t ooff = (size_t)(row0 + tr) * ND + col0 + tc * 16;
#pragma unroll
    for (int q = 0; q < NR; ++q) {
      char* od = (char*)prm.out_all[(self + 1 + q + gi) & 7] + ooff;
#pragma unroll
      for (int ps = 0; ps < 4; ++ps) st16_sys(od + (size_t)ps * 16 * ND, o4[ps]);
    }
  }
  TS(4);
  WAITV(0);
  __syncthreads();
  TS(5);
  if (tid < NR) st4_sys((unsigned*)(WSP(tid) + OFF_OUTF) + gi * NR + self, call);
  if (wv == 0) {
    const unsigned* ofp = (const unsigned*)(ws + OFF_OUTF) + gi * NR + (lane & 7);
    unsigned v = __hip_atomic_load(ofp, __ATOMIC_RELAXED, __HIP_MEMORY_SCOPE_SYSTEM);
    if (!__all(v >= call)) {
      u64 t0 = __builtin_amdgcn_s_memrealtime();
      for (;;) {
        __builtin_amdgcn_s_sleep(1);
        v = __hip_atomic_load(ofp, __ATOMIC_RELAXED, __HIP_MEMORY_SCOPE_SYSTEM);
        if (__all(v >= call)) break;
        if (timed_out(tmo, call, t0)) break;
      }
    }
    if (__hip_atomic_load(tmo, __ATOMIC_RELAXED, __HIP_MEMORY_SCOPE_AGENT) == call && lane == 0) {
      ((signed char*)prm.out_all[self])[(size_t)(ri * TROWS + tm * 64) * ND + cj * TCOLS + tn * 128] = -128;
    }
  }
  TS(6);
}

}

extern "C" void kernel_launch(int rank, int nranks, ncclComm_t comm, hipStream_t stream,
                              void* const* d_in, const int* in_sizes, int n_in, void* d_out,
                              int out_size, const int* in_dtypes, int out_dtype,
                              void* const* d_out_all, void* const* d_ws_all, size_t ws_bytes) {
  (void)comm; (void)d_out; (void)in_dtypes; (void)out_dtype;
  if (nranks != NR || n_in < 2 || in_sizes[0] != MD * KSH || in_sizes[1] != KSH * ND ||
      out_size != MD * ND || ws_bytes < WS_NEED || rank < 0 || rank >= NR) {
    fprintf(stderr, "gemm_ar_q8: unexpected problem shape\n");
    return;
  }
  Params prm;
  prm.x = (const unsigned short*)d_in[0];
  prm.w = (const unsigned short*)d_in[1];
  prm.out_all = d_out_all;
  prm.ws_all = d_ws_all;
  prm.rank = rank;
  gemm_ar_q8_fused<<<GRID, 128, 0, stream>>>(prm);
  hipError_t e = hipGetLastError();
  if (e != hipSuccess) fprintf(stderr, "HIP error %s at %s:%d\n", hipGetErrorString(e), __FILE__, __LINE__);
}
```

```cpp
#include <hip/hip_runtime.h>
#include <rccl/rccl.h>
#include <cstdio>
#include <cstdint>
#include <type_traits>

namespace {

typedef short bf16x8 __attribute__((ext_vector_type(8)));
typedef float f32x4 __attribute__((ext_vector_type(4)));
typedef unsigned v4u __attribute__((ext_vector_type(4)));
typedef unsigned long long u64;

constexpr int NR = 8;
constexpr int MD = 4096, ND = 4096;
constexpr int KSH = 512;
constexpr int TROWS = 1024, TCOLS = 2048;
constexpr int NC = 8;
constexpr int NCOMM = 64;
constexpr int NGEMM = 128;
constexpr int DEPTH = 2;
constexpr int STAGE_B = 32768;
constexpr int GRID = NCOMM + NGEMM;
constexpr int NTICK = 16;
constexpr int WG_PER_TICK = GRID / NTICK;

constexpr size_t OFF_TICK = 0;
constexpr size_t OFF_TMO = 4096;
constexpr size_t OFF_PACE = 4096 + 128;
constexpr size_t OFF_P1F = 8192;
constexpr size_t OFF_AMX = OFF_P1F + 16384;
constexpr size_t OFF_OUTF = OFF_AMX + 8192;
constexpr size_t OFF_A = 1u << 20;
constexpr size_t OFF_B = OFF_A + (8u << 20);
constexpr size_t WS_NEED = OFF_B + (16u << 20);

constexpr u64 DEADLINE_TICKS = 100000000ull;

struct Params {
  const unsigned short* x;
  const unsigned short* w;
  void* const* out_all;
  void* const* ws_all;
  int rank;
};

__device__ __forceinline__ void st16_sys(void* p, v4u v) {
  asm volatile("global_store_dwordx4 %0, %1, off sc0 sc1\n\ts_nop 2" ::"v"(p), "v"(v) : "memory");
}
__device__ __forceinline__ void st4_sys(void* p, unsigned v) {
  __hip_atomic_store((unsigned*)p, v, __ATOMIC_RELAXED, __HIP_MEMORY_SCOPE_SYSTEM);
}
#define TS(i) do {} while (0)
#define WAITV(n) asm volatile("s_waitcnt vmcnt(" #n ")" ::: "memory")

__device__ __forceinline__ bool timed_out(unsigned* tmo, unsigned call, u64 t0) {
  if (__hip_atomic_load(tmo, __ATOMIC_RELAXED, __HIP_MEMORY_SCOPE_AGENT) == call ||
      __builtin_amdgcn_s_memrealtime() - t0 > DEADLINE_TICKS) {
    __hip_atomic_store(tmo, call, __ATOMIC_RELAXED, __HIP_MEMORY_SCOPE_AGENT);
    return true;
  }
  return false;
}

__device__ __forceinline__ void wait_chunk(char* ws, int c, unsigned call, int lane, int fidx) {
  const unsigned* fp = (const unsigned*)(ws + OFF_P1F) + (((lane >> 2) & 7) * NC + c) * NCOMM + fidx;
  unsigned v = __hip_atomic_load(fp, __ATOMIC_RELAXED, __HIP_MEMORY_SCOPE_SYSTEM);
  if (!__all(v >= call)) {
    unsigned* tmo = (unsigned*)(ws + OFF_TMO);
    u64 t0 = __builtin_amdgcn_s_memrealtime();
    for (;;) {
      __builtin_amdgcn_s_sleep(1);
      v = __hip_atomic_load(fp, __ATOMIC_RELAXED, __HIP_MEMORY_SCOPE_SYSTEM);
      if (__all(v >= call)) break;
      if (timed_out(tmo, call, t0)) break;
    }
  }
  asm volatile("" ::: "memory");
}

__global__ __launch_bounds__(256) void gemm_ar_q8_fused(Params prm) {
  __shared__ __attribute__((aligned(1024))) unsigned char smem[3 * STAGE_B + 64];
  const int tid = threadIdx.x, lane = tid & 63, wv = tid >> 6;
  const int self = prm.rank;
#define WSP(p) ((char*)prm.ws_all[(p)])
  char* ws = WSP(self);

  unsigned* sh_u = (unsigned*)(smem + 3 * STAGE_B);
  if (tid == 0) {
    unsigned t = __hip_atomic_fetch_add((unsigned*)(ws + OFF_TICK + (blockIdx.x % NTICK) * 128), 1u,
                                        __ATOMIC_RELAXED, __HIP_MEMORY_SCOPE_AGENT);
    sh_u[0] = t / WG_PER_TICK + 1;
  }
  if (tid >= 1 && tid <= 10) sh_u[tid] = 0;
  __syncthreads();
  const unsigned call = sh_u[0];
  TS(0);
  unsigned* tmo = (unsigned*)(ws + OFF_TMO);

  if (blockIdx.x < NCOMM) {
    const int v = blockIdx.x * 4 + wv;
    const int g = lane >> 4, r = lane & 15;
    v4u xr[NC][2];
    unsigned wr_[NC][2][4];
    const unsigned short* xs = prm.x + (size_t)(16 * v + r) * KSH + 8 * g;
    const unsigned short* wsrc = prm.w + (size_t)(8 * g) * ND + 16 * v + r;
#pragma unroll
    for (int c = 0; c < NC; ++c) {
#pragma unroll
      for (int ks = 0; ks < 2; ++ks) {
        xr[c][ks] = *(const v4u*)(xs + 64 * c + 32 * ks);
#pragma unroll
        for (int jj = 0; jj < 4; ++jj) {
          unsigned lo = wsrc[(size_t)(64 * c + 32 * ks + 2 * jj) * ND];
          unsigned hi = wsrc[(size_t)(64 * c + 32 * ks + 2 * jj + 1) * ND];
          wr_[c][ks][jj] = lo | (hi << 16);
        }
      }
    }
    const int di = v >> 6, dj = v >> 7;
    char* xd[2];
    char* wd[4];
#pragma unroll
    for (int q = 0; q < 2; ++q) xd[q] = WSP(2 * di + ((q + v) & 1)) + OFF_A + (size_t)(v & 63) * 1024 + lane * 16;
#pragma unroll
    for (int q = 0; q < 4; ++q) wd[q] = WSP(dj + 2 * ((q + v) & 3)) + OFF_B + (size_t)(v & 127) * 1024 + lane * 16;
    char* fl = WSP(lane & 7);
    unsigned* flbase = (unsigned*)(fl + OFF_P1F) + (size_t)self * NC * NCOMM + blockIdx.x;
#pragma unroll
    for (int c = 0; c < NC; ++c) {
      if (c >= 2) {
        const unsigned* pc = (const unsigned*)(ws + OFF_PACE + (c - 2) * 128);
        unsigned pv = __hip_atomic_load(pc, __ATOMIC_RELAXED, __HIP_MEMORY_SCOPE_AGENT);
        if (pv < call * NCOMM) {
          u64 t0 = __builtin_amdgcn_s_memrealtime();
          for (;;) {
            __builtin_amdgcn_s_sleep(1);
            pv = __hip_atomic_load(pc, __ATOMIC_RELAXED, __HIP_MEMORY_SCOPE_AGENT);
            if (pv >= call * NCOMM) break;
            if (timed_out(tmo, call, t0)) break;
          }
        }
      }
#pragma unroll
      for (int ks = 0; ks < 2; ++ks) {
        const size_t oa = (size_t)((self * NC + c) * 2 + ks) * 64 * 1024;
        const size_t ob = (size_t)((self * NC + c) * 2 + ks) * 128 * 1024;
        v4u wv4 = {wr_[c][ks][0], wr_[c][ks][1], wr_[c][ks][2], wr_[c][ks][3]};
#pragma unroll
        for (int q = 0; q < 4; ++q) st16_sys(wd[q] + ob, wv4);
#pragma unroll
        for (int q = 0; q < 2; ++q) st16_sys(xd[q] + oa, xr[c][ks]);
      }
      WAITV(0);
      unsigned old = 0;
      if (lane == 0) old = atomicAdd(&sh_u[c + 1], 1u);
      old = __builtin_amdgcn_readfirstlane(old);
      if (old == 3) {
        if (lane < NR) st4_sys(flbase + (size_t)c * NCOMM, call);
        if (lane == 0)
          __hip_atomic_fetch_add((unsigned*)(ws + OFF_PACE + c * 128), 1u, __ATOMIC_RELAXED, __HIP_MEMORY_SCOPE_AGENT);
      }
      if (c == 0) TS(1);
    }
    TS(2);
    return;
  }

  const int gi = blockIdx.x - NCOMM;
  const int xcd = gi & 7, sub = gi >> 3;
  const int tm = (xcd >> 2) * 4 + (sub & 3);
  const int tn = (xcd & 3) * 4 + (sub >> 2);
  const int ri = self >> 1, cj = self & 1;
  const int wr = wv >> 1, wc = wv & 1;

  __builtin_amdgcn_fence(__ATOMIC_ACQUIRE, "");

  f32x4 acc[4][4];
#pragma unroll
  for (int a = 0; a < 4; ++a)
#pragma unroll
    for (int b = 0; b < 4; ++b) acc[a][b] = (f32x4){0.f, 0.f, 0.f, 0.f};

  const int fidx = ((lane & 2) == 0) ? (ri * 16 + tm * 2 + (lane & 1)) : (cj * 32 + tn * 2 + (lane & 1));

  const char* gbase;
  size_t gstride;
  {
    const int half = wv >> 1;
    const int ks = wv & 1;
    gbase = half == 0 ? ws + OFF_A + (size_t)(ks * 64 + tm * 8) * 1024 : ws + OFF_B + (size_t)(ks * 128 + tn * 8) * 1024;
    gbase += lane * 16;
    gstride = half == 0 ? 2 * 64 * 1024 : 2 * 128 * 1024;
  }
  v4u stg[2][8];
  bf16x8 FA[2][4], FB[2][4];
  auto gload = [&](int t, auto par) {
    const int p = (self + (t & 7)) & 7, c = t >> 3;
    const char* g = gbase + (size_t)(p * NC + c) * gstride;
#pragma unroll
    for (int i = 0; i < 8; ++i) stg[decltype(par)::value][i] = *(const v4u*)(g + i * 1024);
  };
  auto lwrite = [&](auto par) {
    unsigned char* slot = smem + decltype(par)::value * STAGE_B + wv * 8192 + lane * 16;
#pragma unroll
    for (int i = 0; i < 8; ++i) *(v4u*)(slot + i * 1024) = stg[decltype(par)::value][i];
  };
  auto half = [&](const unsigned char* rd, int ksr, int fm, auto do_mma) {
    const int fr = fm ^ 1;
#pragma unroll
    for (int nt = 0; nt < 4; ++nt) {
      if constexpr (decltype(do_mma)::value) {
#pragma unroll
        for (int mt = 0; mt < 4; ++mt)
          acc[nt][mt] = __builtin_amdgcn_mfma_f32_16x16x32_bf16(FB[fm][nt], FA[fm][mt], acc[nt][mt], 0, 0, 0);
      }
      FB[fr][nt] = *(const bf16x8*)(rd + (16 + ksr * 8 + wc * 4 + nt) * 1024);
      FA[fr][nt] = *(const bf16x8*)(rd + (ksr * 8 + wr * 4 + nt) * 1024);
      __builtin_amdgcn_sched_barrier(0);
    }
  };
  auto step = [&](int s, auto par, auto first, auto do_write, auto do_load) {
    constexpr int P = decltype(par)::value;
    using NP = std::integral_constant<int, P ^ 1>;
    __builtin_amdgcn_s_barrier();
    const unsigned char* rd = smem + P * STAGE_B + lane * 16;
    half(rd, 0, 1, std::integral_constant<bool, !decltype(first)::value>{});
    if constexpr (decltype(do_write)::value) {
      if constexpr (decltype(do_load)::value) WAITV(8); else WAITV(0);
      lwrite(NP{});
    }
    if constexpr (decltype(do_load)::value) {
      const int sn = s + 3;
      if ((sn & 7) == 0) { TS(8 + (sn >> 3)); wait_chunk(ws, sn >> 3, call, lane, fidx); TS(16 + (sn >> 3)); }
      gload(sn, NP{});
    }
    __builtin_amdgcn_sched_barrier(0);
    half(rd, 1, 0, std::true_type{});
  };
  using T_ = std::true_type;
  using F_ = std::false_type;
  wait_chunk(ws, 0, call, lane, fidx);
  TS(1);
  using P0 = std::integral_constant<int, 0>;
  using P1 = std::integral_constant<int, 1>;
  gload(0, P0{});
  gload(1, P1{});
  WAITV(8);
  lwrite(P0{});
  gload(2, P0{});
  step(0, P0{}, T_{}, T_{}, T_{});
  for (int s = 1; s < 61; s += 2) {
    step(s, P1{}, F_{}, T_{}, T_{});
    step(s + 1, P0{}, F_{}, T_{}, T_{});
  }
  step(61, P1{}, F_{}, T_{}, F_{});
  step(62, P0{}, F_{}, T_{}, F_{});
  step(63, P1{}, F_{}, F_{}, F_{});
#pragma unroll
  for (int nt = 0; nt < 4; ++nt)
#pragma unroll
    for (int mt = 0; mt < 4; ++mt)
      acc[nt][mt] = __builtin_amdgcn_mfma_f32_16x16x32_bf16(FB[1][nt], FA[1][mt], acc[nt][mt], 0, 0, 0);
  __builtin_amdgcn_s_barrier();

  TS(2);
  float am = 0.f;
#pragma unroll
  for (int a = 0; a < 4; ++a)
#pragma unroll
    for (int b = 0; b < 4; ++b)
#pragma unroll
      for (int e = 0; e < 4; ++e) am = fmaxf(am, fabsf(acc[a][b][e]));
#pragma unroll
  for (int o = 32; o > 0; o >>= 1) am = fmaxf(am, __shfl_xor(am, o));
  float* sh_f = (float*)(sh_u + 10);
  if (lane == 0) sh_f[wv] = am;
  __syncthreads();
  if (wv == 0) {
    const float wam = fmaxf(fmaxf(sh_f[0], sh_f[1]), fmaxf(sh_f[2], sh_f[3]));
    if (lane < NR) {
      u64 gr = ((u64)call << 32) | (u64)__float_as_uint(wam);
      __hip_atomic_store((u64*)(WSP(lane) + OFF_AMX) + (self * NGEMM + gi), gr, __ATOMIC_RELAXED,
                         __HIP_MEMORY_SCOPE_SYSTEM);
    }
    const u64* gp = (const u64*)(ws + OFF_AMX) + lane;
    float gm = 0.f;
    u64 t0 = __builtin_amdgcn_s_memrealtime();
    for (;;) {
      u64 v[NR * NGEMM / 64];
#pragma unroll
      for (int k = 0; k < NR * NGEMM / 64; ++k)
        v[k] = __hip_atomic_load(gp + k * 64, __ATOMIC_RELAXED, __HIP_MEMORY_SCOPE_SYSTEM);
      bool ok = true;
      gm = 0.f;
#pragma unroll
      for (int k = 0; k < NR * NGEMM / 64; ++k) {
        ok = ok && ((unsigned)(v[k] >> 32) == call);
        gm = fmaxf(gm, __uint_as_float((unsigned)v[k]));
      }
      if (__all(ok)) break;
      if (timed_out(tmo, call, t0)) break;
      __builtin_amdgcn_s_sleep(1);
    }
#pragma unroll
    for (int o = 32; o > 0; o >>= 1) gm = fmaxf(gm, __shfl_xor(gm, o));
    if (lane == 0) sh_f[4] = gm;
  }
  __syncthreads();
  TS(3);
  const float amax = sh_f[4];
  const bool bad = (__hip_atomic_load(tmo, __ATOMIC_RELAXED, __HIP_MEMORY_SCOPE_AGENT) == call);
  const float scale = amax / 127.0f;

  {
    const int g = lane >> 4, mm = lane & 15;
#pragma unroll
    for (int nt = 0; nt < 4; ++nt)
#pragma unroll
      for (int mt = 0; mt < 4; ++mt) {
        unsigned pk = 0;
#pragma unroll
        for (int e = 0; e < 4; ++e) {
          float qf = (scale > 0.f) ? rintf(acc[nt][mt][e] / scale) : 0.f;
          qf = fminf(fmaxf(qf, -127.f), 127.f);
          int qi = bad ? -128 : (int)qf;
          pk |= ((unsigned)qi & 0xffu) << (8 * e);
        }
        *(unsigned*)(smem + (wr * 64 + mt * 16 + mm) * 144 + wc * 64 + nt * 16 + g * 4) = pk;
      }
  }
  __syncthreads();
  {
    const int row0 = ri * TROWS + tm * 128, col0 = cj * TCOLS + tn * 128;
    const int tr = tid >> 3, tc = tid & 7;
    v4u o4[4];
#pragma unroll
    for (int ps = 0; ps < 4; ++ps) o4[ps] = *(const v4u*)(smem + (ps * 32 + tr) * 144 + tc * 16);
    const size_t ooff = (size_t)(row0 + tr) * ND + col0 + tc * 16;
    char* od[NR];
#pragma unroll
    for (int q = 0; q < NR; ++q) od[q] = (char*)prm.out_all[(self + 1 + q + gi * 4 + wv) & 7] + ooff;
#pragma unroll
    for (int ps = 0; ps < 4; ++ps) {
#pragma unroll
      for (int q = 0; q < NR; ++q) st16_sys(od[q] + (size_t)ps * 32 * ND, o4[ps]);
    }
  }
  TS(4);
  WAITV(0);
  __syncthreads();
  TS(5);
  if (tid < NR) st4_sys((unsigned*)(WSP(tid) + OFF_OUTF) + gi * NR + self, call);
  if (wv == 0) {
    const unsigned* ofp = (const unsigned*)(ws + OFF_OUTF) + gi * NR + (lane & 7);
    unsigned v = __hip_atomic_load(ofp, __ATOMIC_RELAXED, __HIP_MEMORY_SCOPE_SYSTEM);
    if (!__all(v >= call)) {
      u64 t0 = __builtin_amdgcn_s_memrealtime();
      for (;;) {
        __builtin_amdgcn_s_sleep(1);
        v = __hip_atomic_load(ofp, __ATOMIC_RELAXED, __HIP_MEMORY_SCOPE_SYSTEM);
        if (__all(v >= call)) break;
        if (timed_out(tmo, call, t0)) break;
      }
    }
    if (__hip_atomic_load(tmo, __ATOMIC_RELAXED, __HIP_MEMORY_SCOPE_AGENT) == call && lane == 0) {
      ((signed char*)prm.out_all[self])[(size_t)(ri * TROWS + tm * 128) * ND + cj * TCOLS + tn * 128] = -128;
    }
  }
  TS(6);
}

}

extern "C" void kernel_launch(int rank, int nranks, ncclComm_t comm, hipStream_t stream,
                              void* const* d_in, const int* in_sizes, int n_in, void* d_out,
                              int out_size, const int* in_dtypes, int out_dtype,
                              void* const* d_out_all, void* const* d_ws_all, size_t ws_bytes) {
  (void)comm; (void)d_out; (void)in_dtypes; (void)out_dtype;
  if (nranks != NR || n_in < 2 || in_sizes[0] != MD * KSH || in_sizes[1] != KSH * ND ||
      out_size != MD * ND || ws_bytes < WS_NEED || rank < 0 || rank >= NR) {
    fprintf(stderr, "gemm_ar_q8: unexpected problem shape\n");
    return;
  }
  Params prm;
  prm.x = (const unsigned short*)d_in[0];
  prm.w = (const unsigned short*)d_in[1];
  prm.out_all = d_out_all;
  prm.ws_all = d_ws_all;
  prm.rank = rank;
  gemm_ar_q8_fused<<<GRID, 256, 0, stream>>>(prm);
  hipError_t e = hipGetLastError();
  if (e != hipSuccess) fprintf(stderr, "HIP error %s at %s:%d\n", hipGetErrorString(e), __FILE__, __LINE__);
}
```
